```python
import numpy as np

kernel_name = "gpu_a2a_gemm_m4096_k4096_n16384_f32_q8"


ABSMAX_RTOL = 0.02

COMPUTE_DTYPE = "float32"
EPILOGUE = "quant_int8"
IN_QUANT = None
OUT_QUANT = "int8"
IN_DTYPES = {"x": "float32", "w_mat": "float32"}
WORLD_SIZE = 8

def setup_inputs():

    x = np.random.randn(4096, 4096).astype(np.float32)
    w_mat = (np.random.randn(4096, 16384) / np.sqrt(4096)).astype(np.float32)
    return {"x": x, "w_mat": w_mat}

def _quant_dequant(y, amax):
    scale = amax / 127.0
    q = np.clip(np.round(y / scale), -127.0, 127.0)
    return (q * scale).astype(np.float32)
def _epilogue(y):
    return _quant_dequant(y, np.abs(y).max())

def reference(x, w_mat):
    y = (x @ w_mat).astype(np.float32)
    return _epilogue(y)
```

```cpp
#include <hip/hip_runtime.h>
#include <rccl/rccl.h>
#include <cstdio>
#pragma clang diagnostic ignored "-Wint-to-pointer-cast"

#define HIP_CHECK(expr) do { hipError_t _e = (expr); if (_e != hipSuccess) { \
    fprintf(stderr, "HIP error %s at %s:%d\n", hipGetErrorString(_e), __FILE__, __LINE__); \
    return; } } while (0)

#define LAS __attribute__((address_space(3)))
#define GAS __attribute__((address_space(1)))
typedef unsigned u32;
typedef unsigned long long u64;
typedef u32 u32x4 __attribute__((ext_vector_type(4)));
typedef u32 u32x2 __attribute__((ext_vector_type(2)));
typedef float f32x4 __attribute__((ext_vector_type(4)));
typedef float f32x2 __attribute__((ext_vector_type(2)));
typedef short s16x4 __attribute__((ext_vector_type(4)));
typedef __bf16 bf16x2 __attribute__((ext_vector_type(2)));
typedef __bf16 bf16x8 __attribute__((ext_vector_type(8)));
typedef __amdgpu_buffer_rsrc_t rsrc_t;

constexpr int NRANK = 8, NWG = 256, NTHR = 512;
constexpr int NCOL = 16384, KALL = 4096, MROW = 512, KLOC = 512;
constexpr u32 OFF_CNT   = 0;
constexpr u32 OFF_TMO   = 16384;
constexpr u32 OFF_XFLAG = 32768;
constexpr u32 OFF_AMAX  = 65536;
constexpr u32 OFF_PFLAG = 131072;
constexpr u32 OFF_DBG   = 512 * 1024;
constexpr u32 OFF_XS    = 1u << 20;
constexpr u32 OFF_PART  = 8u << 20;
constexpr u32 WS_USED   = OFF_PART + (32u << 20);
constexpr u32 OOB       = 0x80000000u;
constexpr u64 DEADLINE  = 40000000ull;

constexpr int ROWB = 264, SLOT = 16 * ROWB, RING = 12;
constexpr int LDS_PARK = RING * SLOT;
constexpr int LDS_MISC = LDS_PARK + 16384;
constexpr int LDS_BYTES = 96 * 1024;
static_assert(LDS_MISC + 256 <= LDS_BYTES && LDS_PARK % 16 == 0, "lds");

constexpr int AUX_NT = 2, AUX_SC1 = 16, AUX_SYS = 17;
#define W_AUX AUX_NT

struct Params {
    const float* x;
    const float* w;
    char* out;
    int rank;
    int pad;
    char* ws[NRANK];
};

__device__ __forceinline__ rsrc_t mk_rsrc(const void* p, u32 bytes) {
    return __builtin_amdgcn_make_buffer_rsrc((void*)p, (short)0, bytes, 0x00020000);
}
__device__ __forceinline__ u32 cvt2(u32 a, u32 b) {
    f32x2 f = {__builtin_bit_cast(float, a), __builtin_bit_cast(float, b)};
    return __builtin_bit_cast(u32, __builtin_convertvector(f, bf16x2));
}
__device__ __forceinline__ u32x2 trread(u32 addr) {
    return __builtin_bit_cast(u32x2, __builtin_amdgcn_ds_read_tr16_b64_v4i16((LAS s16x4*)addr));
}
__device__ __forceinline__ void mfma(f32x4& c, u32x2 lo, u32x2 hi, u32x4 b) {
    u32x4 a = {lo.x, lo.y, hi.x, hi.y};
    asm volatile("v_mfma_f32_16x16x32_bf16 %0, %1, %2, %0" : "+a"(c) : "v"(a), "v"(b));
}
#define OPAQUE(x) asm volatile("" : "+v"(x))
template <int CTRL> __device__ __forceinline__ u32 dpp(u32 v) {
    return (u32)__builtin_amdgcn_update_dpp(0, (int)v, CTRL, 0xf, 0xf, true);
}
__device__ __forceinline__ u32 wave_max(u32 v) {
    v = max(v, dpp<0xB1>(v)); v = max(v, dpp<0x4E>(v)); v = max(v, dpp<0x141>(v)); v = max(v, dpp<0x140>(v));
    return max(max((u32)__builtin_amdgcn_readlane(v, 0), (u32)__builtin_amdgcn_readlane(v, 16)),
               max((u32)__builtin_amdgcn_readlane(v, 32), (u32)__builtin_amdgcn_readlane(v, 48)));
}
__device__ __forceinline__ u64 now() {
    u64 t = __builtin_amdgcn_s_memrealtime();
    asm volatile("" :: "s"(t));
    return t;
}
#define SCHED() __builtin_amdgcn_sched_barrier(0)
#define BARRIER() do { SCHED(); asm volatile("s_barrier" ::: "memory"); SCHED(); } while (0)
__device__ __forceinline__ void st128x2_sys(char* base, u32 off, u32x4 a, u32x4 c) {
    asm volatile("s_nop 4\n\tglobal_store_dwordx4 %0, %1, %3 sc0 sc1\n\t"
                 "global_store_dwordx4 %0, %2, %3 offset:1024 sc0 sc1\n\ts_nop 1"
                 :: "v"(off), "v"(a), "v"(c), "s"(base) : "memory");
}
__device__ __forceinline__ void st64_sys(char* p, u32 lo, u32 hi) {
    __hip_atomic_store((GAS u64*)p, ((u64)hi << 32) | lo, __ATOMIC_RELAXED, __HIP_MEMORY_SCOPE_SYSTEM);
}
__device__ __forceinline__ bool give_up(char* ws, u32 call, u64 t0) {
    GAS u32* tmo = (GAS u32*)(ws + OFF_TMO);
    if (__hip_atomic_load(tmo, __ATOMIC_RELAXED, __HIP_MEMORY_SCOPE_AGENT) == call ||
        __builtin_amdgcn_s_memrealtime() - t0 > DEADLINE) {
        __hip_atomic_store(tmo, call, __ATOMIC_RELAXED, __HIP_MEMORY_SCOPE_AGENT);
        return true;
    }
    return false;
}
#define STAMP(i) do {} while (0)
#define DBGVAL(i, v) do {} while (0)

__global__ void __launch_bounds__(NTHR, 2) a2a_gemm_q8(Params P) {
    extern __shared__ __attribute__((aligned(16))) char smem_[];
    const u32 lds0 = (u32)(size_t)(LAS char*)smem_;
    int tid = threadIdx.x, lane = tid & 63;
    const int wv = __builtin_amdgcn_readfirstlane(tid >> 6);
    const int b = blockIdx.x, rank = P.rank;
    char* ws = P.ws[0];
    STAMP(0);

    const int label = b & 7, bi = b >> 3, half = label >> 2, lq = label & 3, sp = lq >> 1;
    const int sidx = bi * 2 + (lq & 1);
    const int nt = (sidx >> 4) * 32 + sp * 16 + (sidx & 15);
    int drel, pstep, ph, send_p;
    if (b < 32) { drel = 0; pstep = b >> 1; ph = b & 1; send_p = -1; }
    else { int u = b - 32, j = u / 28, v = u - j * 28;
           drel = 1 + (v >> 2); pstep = 2 * j + ((v >> 1) & 1); ph = v & 1; send_p = j - 1; }
    char* wsd = P.ws[drel & 7];
    const int dst = (rank + drel) & 7;

    const rsrc_t rX  = mk_rsrc(P.x, 4096u * KLOC * 4u);
    const rsrc_t rW  = mk_rsrc(P.w, (u32)KALL * NCOL * 4u);
    const rsrc_t rWS = mk_rsrc(ws, WS_USED);

    const u32 call_v = __builtin_amdgcn_raw_buffer_load_b32(rWS, OFF_CNT + b * 64, 0, AUX_SC1);
    const int g = lane >> 4, r16 = lane & 15;
    u32x4 xin[4];
    {
        u32 vo = (u32)(dst * MROW + ph * 256 + wv * 32 + r16) * (KLOC * 4) + pstep * 128 + g * 16;
        xin[0] = __builtin_amdgcn_raw_buffer_load_b128(rX, vo, 0, AUX_NT);
        xin[1] = __builtin_amdgcn_raw_buffer_load_b128(rX, vo + 64, 0, AUX_NT);
        xin[2] = __builtin_amdgcn_raw_buffer_load_b128(rX, vo, 16 * KLOC * 4, AUX_NT);
        xin[3] = __builtin_amdgcn_raw_buffer_load_b128(rX, vo + 64, 16 * KLOC * 4, AUX_NT);
    }
    SCHED();
    const u32 voffW = ((u32)(2 * wv + (lane >> 5)) * NCOL + nt * 128 + (lane & 31) * 4) * 4u;
    const u32 ks0 = rank * 16 + half * 4;
    u32x4 st[2][3];
#pragma unroll
    for (int i = 0; i < 3; ++i) st[0][i] = __builtin_amdgcn_raw_buffer_load_b128(rW, voffW, (ks0 * 2 + i) << 20, W_AUX);
#pragma unroll
    for (int i = 0; i < 2; ++i) st[1][i] = __builtin_amdgcn_raw_buffer_load_b128(rW, voffW, (ks0 * 2 + 3 + i) << 20, W_AUX);
    SCHED();

    u32 xcc;
    asm volatile("s_getreg_b32 %0, hwreg(HW_REG_XCC_ID)" : "=s"(xcc));
    const u32 iod = (xcc >> 2) & 1;
    if (b < 32) {
        const u32 id = b * 8 + wv, page = id & 127, swap = (id >> 7) ^ ((id >> 1) & 1);
        const char* base = (const char*)P.w + ((size_t)page << 21) + (29u << 16) + ((id >> 7) << 14) + 4096 + 64;
        const char* pa = base + (swap << 13);
        const char* pb = base + ((swap ^ 1) << 13);
        u32 sa, sb, ta, tb;
        asm volatile("s_waitcnt lgkmcnt(0)\n\t"
                     "s_mov_b32 %0, 0x7fc0dead\n\ts_mov_b32 %1, 0x7fc0dead\n\t"
                     "s_load_dword %0, %4, 0x0\n\ts_load_dword %1, %5, 0x0\n\t"
                     "s_waitcnt lgkmcnt(1)\n\t"
                     "s_mov_b32 %2, %0\n\ts_mov_b32 %3, %1\n\t"
                     "s_waitcnt lgkmcnt(0)"
                     : "=&s"(sa), "=&s"(sb), "=&s"(ta), "=&s"(tb) : "s"(pa), "s"(pb) : "memory");
        const bool ga = ta != 0x7fc0deadu, gb = tb != 0x7fc0deadu;
        const u32 nearbit = ga ? swap : (swap ^ 1);
        const u32 vote = (ga == gb) ? 0u : (((nearbit ^ page ^ iod) & 1) ? 0x100u : 1u);
        if (lane == 0) *(volatile LAS u32*)(lds0 + LDS_MISC + wv * 4) = vote;
    }
    SCHED();

    u32x4 xf0 = {cvt2(xin[0].x, xin[0].y), cvt2(xin[0].z, xin[0].w), cvt2(xin[1].x, xin[1].y), cvt2(xin[1].z, xin[1].w)};
    u32x4 xf1 = {cvt2(xin[2].x, xin[2].y), cvt2(xin[2].z, xin[2].w), cvt2(xin[3].x, xin[3].y), cvt2(xin[3].z, xin[3].w)};
    const u32 call = __builtin_amdgcn_readfirstlane(call_v) + 1;
    const u32 piece_off = OFF_XS + (u32)(rank * 16 + pstep) * 32768u + ph * 16384 + wv * 2048;
    const u32 pflag_off = OFF_XFLAG + (u32)(rank * 16 + pstep) * 128u + ph * 8;
    const u32 park = lds0 + LDS_PARK + wv * 2048 + lane * 16;
    if (send_p < 0) {
        st128x2_sys(wsd + piece_off, lane * 16, xf0, xf1);
    } else {
        *(LAS u32x4*)park = xf0;
        *(LAS u32x4*)(park + 1024) = xf1;
    }
    SCHED();
    u32 wr_cur = lds0 + (2 * wv + (lane >> 5)) * ROWB + (lane & 31) * 8, wr_oth = wr_cur + 6 * SLOT;
    u32 rd_cur = lds0 + (4 * g + (r16 >> 2)) * ROWB + (lane & 3) * 64,   rd_oth = rd_cur + 6 * SLOT;
#define WSTORE(addr, v) (*(LAS u32x2*)(addr) = u32x2{cvt2((v).x, (v).y), cvt2((v).z, (v).w)})
#pragma unroll
    for (int i = 0; i < 3; ++i) WSTORE(wr_cur + i * SLOT, st[0][i]);
#pragma unroll
    for (int i = 0; i < 2; ++i) WSTORE(wr_cur + (3 + i) * SLOT, st[1][i]);
    SCHED();
    asm volatile("s_waitcnt vmcnt(0) lgkmcnt(0)" ::: "memory");
    BARRIER();
    if (send_p < 0 && tid == 0) {
        u32 info = iod << 16;
        if (b < 32) for (int i = 0; i < 8; ++i) info += *(volatile LAS u32*)(lds0 + LDS_MISC + i * 4);
        st64_sys(wsd + pflag_off, info, call);
    }
    SCHED();
#pragma unroll
    for (int i = 0; i < 3; ++i) st[1][i] = __builtin_amdgcn_raw_buffer_load_b128(rW, voffW, (ks0 * 2 + 5 + i) << 20, W_AUX);
    SCHED();
    STAMP(1);

    u32 kp;
    {
        const u32 vo = lane < 16 ? OFF_XFLAG + (u32)(rank * 16 + lane) * 128u : OOB;
        const u64 t0 = now();
        u32x4 f;
        for (;;) {
            f = __builtin_amdgcn_raw_buffer_load_b128(rWS, vo, 0, AUX_SYS);
            const bool bad = lane < 16 && (f.y != call || f.w != call);
            if (__builtin_amdgcn_ballot_w64(bad) == 0) break;
            if (give_up(ws, call, t0)) break;
            __builtin_amdgcn_s_sleep(1);
        }
        __builtin_amdgcn_fence(__ATOMIC_ACQUIRE, "wavefront");
        int d = (int)((f.x >> 8) & 255) + (int)((f.z >> 8) & 255) - (int)(f.x & 255) - (int)(f.z & 255);
        d += dpp<0xB1>(d); d += dpp<0x4E>(d); d += dpp<0x141>(d); d += dpp<0x140>(d);
        const int dsum = __builtin_amdgcn_readfirstlane(d);
        const u32 i01 = __builtin_amdgcn_readlane((lq & 1) ? f.z : f.x, lq >> 1);
        u32 par = dsum > 0;
        kp = (par ^ sp ^ (i01 >> 16) ^ half) & 1;
        DBGVAL(10, (u64)(long long)dsum);
        DBGVAL(11, xcc);
    }
    u32 tab;
    {
        const u32 u = lane - 8, j = (u * 37) >> 8, i = u - j * 7;
        tab = lane < 4 ? ks0 + lane : lane < 8 ? rank * 16 + 2 * lane + kp : (((rank + 1 + i) & 7) * 16 + 2 * j + kp);
    }
    u32 voffXk = OFF_XS - 2048 + wv * 4096 + lane * 16 + half * 2048;
    OPAQUE(voffXk);
    const u32 xsd = half ? -2048 : 2048;
#define KOFF(s, sh) (((u32)__builtin_amdgcn_readlane(tab, (s) & 63) << (sh)) | (((u32)(s) >> 6) << 31))
#define XLOAD(dstv, so) do { const u32 so_ = (so) + 2048, sx_ = so_ + xsd; \
        (dstv)[0] = __builtin_amdgcn_raw_buffer_load_b128(rWS, voffXk, so_, AUX_SYS); \
        (dstv)[1] = __builtin_amdgcn_raw_buffer_load_b128(rWS, voffXk + 1024, so_, AUX_SYS); \
        (dstv)[2] = __builtin_amdgcn_raw_buffer_load_b128(rWS, voffXk, sx_, AUX_SYS); \
        (dstv)[3] = __builtin_amdgcn_raw_buffer_load_b128(rWS, voffXk + 1024, sx_, AUX_SYS); } while (0)
    u32x4 xr[3][4];
    const u32 pf_off = OFF_XS + bi * 1024 + wv * 128;
    u32 pfsink = __builtin_amdgcn_raw_buffer_load_b32(rWS, (u32)(lane - 3) < 3u ? tab << 15 : OOB, pf_off, AUX_SYS);
#pragma unroll
    for (int a = 0; a < 3; ++a) XLOAD(xr[a], (ks0 + a) << 15);
    SCHED();
    STAMP(2);

    f32x4 acc[4][8];
#pragma unroll
    for (int i = 0; i < 4; ++i)
#pragma unroll
        for (int t = 0; t < 8; ++t) acc[i][t] = f32x4{0.f, 0.f, 0.f, 0.f};
    constexpr int PF = 3;
    u32x2 fl[4], fh[4];
#define FREAD(n) do { const u32 ba_ = ((n) < 24 ? rd_cur : rd_oth) + ((((n) % 24) >> 3) * 2) * SLOT + ((n) & 7) * 8; \
        fl[(n) & 3] = trread(ba_); fh[(n) & 3] = trread(ba_ + SLOT); } while (0)
#define TILE(n) do { FREAD((n) + PF); SCHED(); \
        _Pragma("unroll") for (int i_ = 0; i_ < 4; ++i_) \
            mfma(acc[i_][(n) & 7], fl[(n) & 3], fh[(n) & 3], xr[(n) >> 3][i_]); \
        SCHED(); } while (0)
#pragma unroll
    for (int n = 0; n < PF; ++n) FREAD(n);
    SCHED();

    for (int p = 0; p < 21; ++p) {
        const int s0 = 3 * p;
        if (p == send_p) {
            const u32 l16 = voffXk & 1023, pk = lds0 + LDS_PARK + wv * 2048 + l16;
            const u32x4 a = *(LAS u32x4*)pk, c = *(LAS u32x4*)(pk + 1024);
            st128x2_sys(wsd + piece_off, l16, a, c);
        } else if (p == send_p + 1 && p > 0 && wv == 0) {
            if ((voffXk & 1023) == 0) st64_sys(wsd + pflag_off, 0, call);
        }
        SCHED();
        {
            const u32 ka = KOFF(s0 + 4, 21), kb = KOFF(s0 + 5, 21);
            st[0][0] = __builtin_amdgcn_raw_buffer_load_b128(rW, voffW, ka, W_AUX);
            st[0][1] = __builtin_amdgcn_raw_buffer_load_b128(rW, voffW, ka + (1u << 20), W_AUX);
            st[0][2] = __builtin_amdgcn_raw_buffer_load_b128(rW, voffW, kb, W_AUX);
        }
        const bool fact = (voffXk & 1023) - (u32)(s0 + 6) * 16 < 48u;
        u32x4 fg = __builtin_amdgcn_raw_buffer_load_b128(rWS, fact ? tab << 7 : OOB, OFF_XFLAG, AUX_SYS);
        SCHED();
        TILE(0); TILE(1); TILE(2); TILE(3); TILE(4); TILE(5); TILE(6); TILE(7);
        XLOAD(xr[0], KOFF(s0 + 3, 15));
        SCHED();
        WSTORE(wr_cur + 5 * SLOT, st[1][0]);
        TILE(8);
        WSTORE(wr_oth, st[1][1]);
        TILE(9);
        WSTORE(wr_oth + SLOT, st[1][2]);
        TILE(10); TILE(11);
        asm volatile("s_waitcnt lgkmcnt(4)" ::: "memory");
        BARRIER();
        {
            const u32 ka = KOFF(s0 + 5, 21), kb = KOFF(s0 + 6, 21);
            st[1][0] = __builtin_amdgcn_raw_buffer_load_b128(rW, voffW, ka + (1u << 20), W_AUX);
            st[1][1] = __builtin_amdgcn_raw_buffer_load_b128(rW, voffW, kb, W_AUX);
            st[1][2] = __builtin_amdgcn_raw_buffer_load_b128(rW, voffW, kb + (1u << 20), W_AUX);
        }
        SCHED();
        if (__builtin_amdgcn_ballot_w64(fact && (fg.y != call || fg.w != call)) != 0) {
            const u64 t0 = now();
            for (;;) {
                fg = __builtin_amdgcn_raw_buffer_load_b128(rWS, fact ? tab << 7 : OOB, OFF_XFLAG, AUX_SYS);
                if (__builtin_amdgcn_ballot_w64(fact && (fg.y != call || fg.w != call)) == 0) break;
                if (give_up(ws, call, t0)) break;
                __builtin_amdgcn_s_sleep(1);
            }
        }
        __builtin_amdgcn_fence(__ATOMIC_ACQUIRE, "wavefront");
        asm volatile("" :: "v"(pfsink));
        pfsink = __builtin_amdgcn_raw_buffer_load_b32(rWS, fact ? tab << 15 : OOB, pf_off, AUX_SYS);
        SCHED();
        TILE(12); TILE(13); TILE(14); TILE(15);
        XLOAD(xr[1], KOFF(s0 + 4, 15));
        SCHED();
        TILE(16); TILE(17); TILE(18); TILE(19);
        WSTORE(wr_oth + 2 * SLOT, st[0][0]);
        TILE(20);
        WSTORE(wr_oth + 3 * SLOT, st[0][1]);
        TILE(21);
        WSTORE(wr_oth + 4 * SLOT, st[0][2]);
        TILE(22); TILE(23);
        XLOAD(xr[2], KOFF(s0 + 5, 15));
        SCHED();
        asm volatile("s_waitcnt vmcnt(16) lgkmcnt(4)" ::: "memory");
        BARRIER();
        { u32 t = rd_cur; rd_cur = rd_oth; rd_oth = t; t = wr_cur; wr_cur = wr_oth; wr_oth = t; }
    }
    TILE(0); TILE(1); TILE(2); TILE(3); TILE(4); TILE(5); TILE(6); TILE(7);
    asm volatile("s_nop 7" :: "v"(pfsink));
    SCHED();
    tid = wv * 64 + ((voffXk & 1023) >> 4); lane = tid & 63;
    STAMP(3);

    {
        const u32 vo = OFF_PART + (u32)b * 131072u + wv * 16384 + lane * 16;
#pragma unroll
        for (int i = 0; i < 2; ++i)
#pragma unroll
            for (int t = 0; t < 8; ++t)
                __builtin_amdgcn_raw_buffer_store_b128(__builtin_bit_cast(u32x4, acc[2 + i][t]), rWS, vo + (t & 3) * 1024, (i * 2 + (t >> 2)) * 4096, AUX_SYS);
        asm volatile("s_waitcnt vmcnt(0)" ::: "memory");
        if (lane == 0)
            __hip_atomic_store((GAS u32*)(ws + OFF_PFLAG + (b * 8 + wv) * 128), call, __ATOMIC_RELAXED, __HIP_MEMORY_SCOPE_SYSTEM);
        STAMP(4);
        GAS u32* pf = (GAS u32*)(ws + OFF_PFLAG + ((b ^ 4) * 8 + wv) * 128);
        const u64 t0 = now();
        while (__hip_atomic_load(pf, __ATOMIC_RELAXED, __HIP_MEMORY_SCOPE_SYSTEM) < call) {
            if (give_up(ws, call, t0)) break;
            __builtin_amdgcn_s_sleep(1);
        }
        __builtin_amdgcn_fence(__ATOMIC_ACQUIRE, "wavefront");
        STAMP(5);
        const u32 vi = OFF_PART + (u32)(b ^ 4) * 131072u + wv * 16384 + lane * 16;
#pragma unroll
        for (int i = 0; i < 2; ++i)
#pragma unroll
            for (int t = 0; t < 8; ++t)
                acc[2 + i][t] = __builtin_bit_cast(f32x4, __builtin_amdgcn_raw_buffer_load_b128(rWS, vi + (t & 3) * 1024, (i * 2 + (t >> 2)) * 4096, AUX_SYS));
        SCHED();
    }
    float am = 0.f;
#pragma unroll
    for (int i = 0; i < 2; ++i)
#pragma unroll
        for (int t = 0; t < 8; ++t) {
            acc[i][t] += acc[2 + i][t];
#pragma unroll
            for (int e = 0; e < 4; ++e) am = fmaxf(am, fabsf(acc[i][t][e]));
        }
    const u32 amw = wave_max(__builtin_bit_cast(u32, am));
    volatile LAS u32* misc = (volatile LAS u32*)(lds0 + LDS_MISC);
    if (lane == 0) misc[16 + wv] = amw;
    asm volatile("s_waitcnt lgkmcnt(0)" ::: "memory");
    BARRIER();
    STAMP(6);
    if (tid < 8) {
        u32 m = 0;
#pragma unroll
        for (int i = 0; i < 8; ++i) m = max(m, misc[16 + i]);
        char* wsl = P.ws[0];
#pragma unroll
        for (int i = 1; i < 8; ++i) wsl = tid == i ? P.ws[i] : wsl;
        st64_sys(wsl + OFF_AMAX + (rank * NWG + b) * 8, m, call);
    }
    u32 gm, bad_call;
    {
        const u64 t0 = now();
        u32x4 a, c;
        for (;;) {
            a = __builtin_amdgcn_raw_buffer_load_b128(rWS, OFF_AMAX + tid * 16, 0, AUX_SYS);
            c = __builtin_amdgcn_raw_buffer_load_b128(rWS, OFF_AMAX + tid * 16, 8192, AUX_SYS);
            bad_call = __builtin_amdgcn_raw_buffer_load_b32(rWS, OFF_TMO, 0, AUX_SC1);
            const bool bad = a.y != call || a.w != call || c.y != call || c.w != call;
            if (__builtin_amdgcn_ballot_w64(bad) == 0) break;
            if (give_up(ws, call, t0)) { bad_call = call; break; }
            __builtin_amdgcn_s_sleep(1);
        }
        gm = wave_max(max(max(a.x, a.z), max(c.x, c.z)));
        if (lane == 0) { misc[32 + wv] = gm; misc[40 + wv] = bad_call == call; }
        asm volatile("s_waitcnt lgkmcnt(0)" ::: "memory");
        BARRIER();
        gm = 0; bad_call = 0;
#pragma unroll
        for (int i = 0; i < 8; ++i) { gm = max(gm, misc[32 + i]); bad_call |= misc[40 + i]; }
    }
    STAMP(7);
    {
        const float amax = __builtin_bit_cast(float, gm);
        const float inv = amax > 0.f ? 127.0f / amax : 0.f;
        const rsrc_t rO = mk_rsrc(P.out, (u32)MROW * NCOL);
#pragma unroll
        for (int i = 0; i < 2; ++i) {
            u32 q[8];
#pragma unroll
            for (int t = 0; t < 8; ++t) {
                u32 e[4];
#pragma unroll
                for (int k = 0; k < 4; ++k) {
                    const float v = __builtin_amdgcn_fmed3f(acc[i][t][k] * inv, -127.f, 127.f);
                    e[k] = __builtin_bit_cast(u32, v + 12582912.0f);
                }
                const u32 lo = __builtin_amdgcn_perm(e[1], e[0], 0x0c0c0400u);
                const u32 hi = __builtin_amdgcn_perm(e[3], e[2], 0x04000c0cu);
                q[t] = bad_call ? 0x80808080u : (lo | hi);
            }
            const u32 vo = (u32)(wv * 64 + (half * 2 + i) * 16 + (lane & 15)) * NCOL + nt * 128 + (lane >> 4) * 32;
            __builtin_amdgcn_raw_buffer_store_b128(u32x4{q[0], q[1], q[2], q[3]}, rO, vo, 0, 0);
            __builtin_amdgcn_raw_buffer_store_b128(u32x4{q[4], q[5], q[6], q[7]}, rO, vo + 16, 0, 0);
        }
    }
    if (tid == 0)
        __hip_atomic_store((GAS u32*)(ws + OFF_CNT + b * 64), call, __ATOMIC_RELAXED, __HIP_MEMORY_SCOPE_AGENT);
    STAMP(8);
}

extern "C" void kernel_launch(int rank, int nranks, ncclComm_t comm,
                              hipStream_t stream,
                              void* const* d_in, const int* in_sizes, int n_in,
                              void* d_out, int out_size,
                              const int* in_dtypes, int out_dtype,
                              void* const* d_out_all,
                              void* const* d_ws_all, size_t ws_bytes) {
    if (nranks != NRANK || (unsigned)rank >= (unsigned)NRANK || ws_bytes < WS_USED) {
        fprintf(stderr, "unexpected configuration\n");
        return;
    }
    HIP_CHECK(hipFuncSetAttribute((const void*)a2a_gemm_q8, hipFuncAttributeMaxDynamicSharedMemorySize, LDS_BYTES));
    Params P;
    P.x = (const float*)d_in[0];
    P.w = (const float*)d_in[1];
    P.out = (char*)d_out;
    for (int p = 0; p < NRANK; ++p) P.ws[p] = (char*)d_ws_all[(rank + p) % NRANK];
    P.rank = rank;
    P.pad = 0;
    hipLaunchKernelGGL(a2a_gemm_q8, dim3(NWG), dim3(NTHR), LDS_BYTES, stream, P);
    HIP_CHECK(hipGetLastError());
}
```

```cpp
#include <hip/hip_runtime.h>
#include <rccl/rccl.h>
#include <cstdio>
#pragma clang diagnostic ignored "-Wint-to-pointer-cast"

#define HIP_CHECK(expr) do { hipError_t _e = (expr); if (_e != hipSuccess) { \
    fprintf(stderr, "HIP error %s at %s:%d\n", hipGetErrorString(_e), __FILE__, __LINE__); \
    return; } } while (0)

#define LAS __attribute__((address_space(3)))
#define GAS __attribute__((address_space(1)))
typedef unsigned u32;
typedef unsigned long long u64;
typedef u32 u32x4 __attribute__((ext_vector_type(4)));
typedef u32 u32x2 __attribute__((ext_vector_type(2)));
typedef float f32x4 __attribute__((ext_vector_type(4)));
typedef float f32x2 __attribute__((ext_vector_type(2)));
typedef short s16x4 __attribute__((ext_vector_type(4)));
typedef __bf16 bf16x2 __attribute__((ext_vector_type(2)));
typedef __amdgpu_buffer_rsrc_t rsrc_t;

constexpr int NRANK = 8, NWG = 256, NTHR = 512;
constexpr int NCOL = 16384, KALL = 4096, MROW = 512, KLOC = 512;
constexpr u32 OFF_CNT   = 0;
constexpr u32 OFF_TMO   = 16384;
constexpr u32 OFF_VOTE  = 16384 + 1024;
constexpr u32 OFF_CLK   = 16384 + 2048;
constexpr u32 OFF_XFLAG = 32768;
constexpr u32 OFF_AMAX  = 65536;
constexpr u32 OFF_GMAX  = 65536 + 4096;
constexpr u32 OFF_PFLAG = 131072;
constexpr u32 OFF_XS    = 1u << 20;
constexpr u32 OFF_PART  = 8u << 20;
constexpr u32 WS_USED   = OFF_PART + (16u << 20);
constexpr u32 OOB       = 0x80000000u;
#define PACE0 308
#define PACE_RAMP 0x121c26323f4c5863ull
#define PACE1 211
#define FAST0 262
#define FAST_RAMP 0x0b0b0f161e262e36ull
#define FAST_CATCH 27
#define CLK_AT 4
#define CLK_THR 175
#define PACE_T0 465
#define ST_BITS "sc0 sc1"
#define X0 4
#define XD 4
#define LEAN 8
#define WIDE 4
#define WSTEP 13
#define WLO(p) ((p) < WIDE ? 3 * (p) + 3 : 3 * WIDE + 3 + ((p) - WIDE) * WSTEP)
#define PL_AT (1 - grp)
#define TOUCH 1
#define FLAG_AT 26
#define POLL_AT 33
#define NXA 10
#define STAGGER 140
#define FAR 150
#define CATCH_FAR 24
#define CATCH (p < 6 ? 28 : p < 14 ? 30 : 28)
constexpr u32 WARM_TICKS = 5000;
constexpr u64 DEADLINE  = 40000000ull;

constexpr int ROWB = 264, SLOT = 16 * ROWB, NBUF = 5, BUF = 6 * SLOT, RING = 6 * NBUF + 2;
#define NF 10
constexpr int NPER = (64 - NF) / 3;
constexpr int LDS_PARK = RING * SLOT;
constexpr int LDS_MISC = LDS_PARK + 16384;
constexpr int LDS_BYTES = 152 * 1024;
static_assert(LDS_MISC + 256 <= LDS_BYTES && LDS_PARK % 16 == 0, "lds");

constexpr int POLLW = 3, GATHER_WG = 250;
constexpr int AUX_NT = 2, AUX_SC1 = 16, AUX_SYS = 17;
#define W_AUX AUX_NT

#define ALLOC_FIRST 3
#define X_AUX AUX_NT
#define PF_AUX AUX_SYS
#define FETCH_AUX AUX_SYS
#define OUT_AUX AUX_NT
#define PART_AUX AUX_SYS
#define AMAX_SLEEP 1
#define PRIO_C 0
#define PRIO_L 1
template <int V> struct IC { static constexpr int value = V; };
struct Params {
    const float* x;
    const float* w;
    char* out;
    int rank;
    int pad;
    char* ws[NRANK];
};

__device__ __forceinline__ rsrc_t mk_rsrc(const void* p, u32 bytes) {
    return __builtin_amdgcn_make_buffer_rsrc((void*)p, (short)0, bytes, 0x00020000);
}
__device__ __forceinline__ u32 cvt2(u32 a, u32 b) {
    f32x2 f = {__builtin_bit_cast(float, a), __builtin_bit_cast(float, b)};
    return __builtin_bit_cast(u32, __builtin_convertvector(f, bf16x2));
}
__device__ __forceinline__ u32x2 trread(u32 addr) {
    return __builtin_bit_cast(u32x2, __builtin_amdgcn_ds_read_tr16_b64_v4i16((LAS s16x4*)addr));
}
__device__ __forceinline__ void mfma(f32x4& c, u32x2 lo, u32x2 hi, u32x4 b) {
    u32x4 a = {lo.x, lo.y, hi.x, hi.y};
    asm volatile("v_mfma_f32_16x16x32_bf16 %0, %1, %2, %0" : "+a"(c) : "v"(a), "v"(b));
}
#define OPAQUE(x) asm volatile("" : "+v"(x))
template <int CTRL> __device__ __forceinline__ u32 dpp(u32 v) {
    return (u32)__builtin_amdgcn_update_dpp(0, (int)v, CTRL, 0xf, 0xf, true);
}
__device__ __forceinline__ u32 wave_max(u32 v) {
    v = max(v, dpp<0xB1>(v)); v = max(v, dpp<0x4E>(v)); v = max(v, dpp<0x141>(v)); v = max(v, dpp<0x140>(v));
    return max(max((u32)__builtin_amdgcn_readlane(v, 0), (u32)__builtin_amdgcn_readlane(v, 16)),
               max((u32)__builtin_amdgcn_readlane(v, 32), (u32)__builtin_amdgcn_readlane(v, 48)));
}
__device__ __forceinline__ u64 now() {
    u64 t = __builtin_amdgcn_s_memrealtime();
    asm volatile("" :: "s"(t));
    return t;
}
#define SCHED() __builtin_amdgcn_sched_barrier(0)
#define BARRIER() do { SCHED(); asm volatile("s_barrier" ::: "memory"); SCHED(); } while (0)
__device__ __forceinline__ void st128x2_sys(char* base, u32 off, u32x4 a, u32x4 c) {
    asm volatile("s_nop 4\n\tglobal_store_dwordx4 %0, %1, %3 " ST_BITS "\n\t"
                 "global_store_dwordx4 %0, %2, %3 offset:1024 " ST_BITS "\n\ts_nop 1"
                 :: "v"(off), "v"(a), "v"(c), "s"(base) : "memory");
}
__device__ __forceinline__ void st32_sys(char* p, u32 v) {
    __hip_atomic_store((GAS u32*)p, v, __ATOMIC_RELAXED, __HIP_MEMORY_SCOPE_SYSTEM);
}
__device__ __forceinline__ void st64_sys(char* p, u32 lo, u32 hi) {
    __hip_atomic_store((GAS u64*)p, ((u64)hi << 32) | lo, __ATOMIC_RELAXED, __HIP_MEMORY_SCOPE_SYSTEM);
}
__device__ __forceinline__ bool give_up(char* ws, u32 call, u64 t0) {
    GAS u32* tmo = (GAS u32*)(ws + OFF_TMO);
    if (__hip_atomic_load(tmo, __ATOMIC_RELAXED, __HIP_MEMORY_SCOPE_AGENT) == call ||
        __builtin_amdgcn_s_memrealtime() - t0 > DEADLINE) {
        __hip_atomic_store(tmo, call, __ATOMIC_RELAXED, __HIP_MEMORY_SCOPE_AGENT);
        return true;
    }
    return false;
}
__device__ __forceinline__ u32 race(const char* base, u32 swap, u32 flip) {
    const char* pa = base + (swap << 13);
    const char* pb = base + ((swap ^ 1) << 13);
    u32 sa, sb, ta, tb;
    asm volatile("s_waitcnt lgkmcnt(0)\n\t"
                 "s_mov_b32 %0, 0x7fc0dead\n\ts_mov_b32 %1, 0x7fc0dead\n\t"
                 "s_load_dword %0, %4, 0x0\n\ts_load_dword %1, %5, 0x0\n\t"
                 "s_waitcnt lgkmcnt(1)\n\t"
                 "s_mov_b32 %2, %0\n\ts_mov_b32 %3, %1\n\t"
                 "s_waitcnt lgkmcnt(0)"
                 : "=&s"(sa), "=&s"(sb), "=&s"(ta), "=&s"(tb) : "s"(pa), "s"(pb) : "memory");
    const bool ga = ta != 0x7fc0deadu, gb = tb != 0x7fc0deadu;
    return ga == gb ? 1u : (((ga ? swap : swap ^ 1) ^ flip) & 1) * 2;
}

__global__ void __launch_bounds__(NTHR, 2) a2a_gemm_q8(Params P) {
    extern __shared__ __attribute__((aligned(16))) char smem_[];
    const u32 lds0 = (u32)(size_t)(LAS char*)smem_;
    int tid = threadIdx.x, lane = tid & 63;
    const int wv = __builtin_amdgcn_readfirstlane(tid >> 6);
    const int b = blockIdx.x, rank = P.rank;
    char* ws = P.ws[0];
    const u32 t_start = (u32)__builtin_amdgcn_s_memrealtime();

    const int label = b & 7, bi = b >> 3, half = label >> 2, lq = label & 3, sp = lq >> 1;
    const int sidx = bi * 2 + (lq & 1);
    const int nt = (sidx >> 4) * 32 + sp * 16 + (sidx & 15);
    int drel, pstep, ph, send_p;
    if (b < 32) { drel = 0; pstep = b >> 1; ph = b & 1; send_p = -2; }
    else { const int j = (label + bi) & 7, v = bi - 4, q = v / 7;
           drel = 1 + v - q * 7; pstep = 2 * j + (q >> 1); ph = q & 1; send_p = j ? (j - 1) / 2 : -2; }
    char* wsd = P.ws[drel & 7];
    const int dst = (rank + drel) & 7;
    asm volatile("" :: "s"(P.x), "s"(P.w), "s"(wsd), "s"(ws), "s"(rank));

    const rsrc_t rX  = mk_rsrc(P.x, 4096u * KLOC * 4u);
    const rsrc_t rW  = mk_rsrc(P.w, (u32)KALL * NCOL * 4u);
    const rsrc_t rWS = mk_rsrc(ws, WS_USED);

    const u32x2 cnt_v = __builtin_amdgcn_raw_buffer_load_b64(rWS, OFF_CNT + b * 64, 0, AUX_SC1);
    const int g = lane >> 4, r16 = lane & 15, grp = wv >> 2, wj = wv & 3;
    const u32 ks0 = rank * 16 + half * 8;
    const u32 vox = (u32)(dst * MROW + ph * 256 + wv * 32 + (lane >> 3)) * (KLOC * 4) + pstep * 128 + (lane & 7) * 16;
    const u32 piece_off = OFF_XS + (u32)(rank * 16 + pstep) * 32768u + ph * 16384;
    const u32 pflag_off = OFF_XFLAG + (u32)(rank * 16 + pstep) * 128u + ph * 4;
    volatile LAS u32* misc = (volatile LAS u32*)(lds0 + LDS_MISC);
#define XIN_LOAD() do { _Pragma("unroll") for (int i = 0; i < 4; ++i) \
        xin[i] = __builtin_amdgcn_raw_buffer_load_b128(rX, vox, i * 8 * KLOC * 4, X_AUX); SCHED(); } while (0)
#define XIN_PARK() do { const u32 pa_ = lds0 + LDS_PARK + wv * 2048 + (lane & 3) * 256 + (lane >> 3) * 16 + (lane & 4) * 2; \
        _Pragma("unroll") for (int i = 0; i < 4; ++i) \
            *(LAS u32x2*)(pa_ + (i >> 1) * 1024 + (i & 1) * 128) = u32x2{cvt2(xin[i].x, xin[i].y), cvt2(xin[i].z, xin[i].w)}; } while (0)
    const u32 park = lds0 + LDS_PARK + wv * 2048 + lane * 16;
    if (send_p == -2) {
        u32x4 xin[4];
        XIN_LOAD();
        XIN_PARK();
        asm volatile("" ::: "memory");
        st128x2_sys(wsd + piece_off + wv * 2048, lane * 16, *(LAS u32x4*)park, *(LAS u32x4*)(park + 1024));
        SCHED();
        asm volatile("s_waitcnt vmcnt(0)" ::: "memory");
        BARRIER();
        if (tid == 0) st32_sys(wsd + pflag_off, cnt_v.x + 1);
        SCHED();
    }
    u32 xcc;
    asm volatile("s_getreg_b32 %0, hwreg(HW_REG_XCC_ID)" : "=s"(xcc));
    if (b >= 128 && wv == 7) {
        const u32 iod = (xcc >> 2) & 1, swap = ((b >> 1) ^ (b >> 3)) & 1;
        const u32 line = ((t_start + b * 40503u) * 2654435761u >> 11) & 0x1fdf80u;
        const u32 vw = race((const char*)P.w + ((size_t)(b & 127) << 21) + line, swap, b ^ iod);
        const u32 vp = race(ws + WS_USED + ((size_t)(b & 7) << 21) + line, swap, b ^ iod);
        if (lane == 0) st64_sys(ws + OFF_VOTE + (b & 127) * 8, vw | vp << 10 | iod << 20, cnt_v.x + 1);
    }
    SCHED();
    const u32 voffW = ((u32)(2 * wj + (lane >> 5)) * NCOL + nt * 128 + (lane & 31) * 4) * 4u;
    const u32 fvo = lane < 3 && wv == POLLW ? OFF_XFLAG + (ks0 + lane) * 128u : OOB;
    u32x4 st[6], xin[4];
#pragma unroll
    for (int i = 0; i < 6; ++i) st[i] = __builtin_amdgcn_raw_buffer_load_b128(rW, voffW, (ks0 * 4 + grp * 6 + i) << 19, W_AUX);
    SCHED();
    if (send_p >= 0) XIN_LOAD();
    u32x2 f0 = __builtin_amdgcn_raw_buffer_load_b64(rWS, fvo, 0, AUX_SYS);
    SCHED();
    f32x4 acc[4][8];
#pragma unroll
    for (int i = 0; i < 4; ++i)
#pragma unroll
        for (int t = 0; t < 8; ++t) { acc[i][t] = f32x4{0.f, 0.f, 0.f, 0.f}; asm volatile("" : "+a"(acc[i][t])); }
    const u32 call = __builtin_amdgcn_readfirstlane(cnt_v.x) + 1;
    const bool warm = call > 1 && (u32)now() - (u32)__builtin_amdgcn_readfirstlane(cnt_v.y) < WARM_TICKS;
    u32 wr0 = lds0 + grp * 3 * SLOT + (2 * wj + (lane >> 5)) * ROWB + (lane & 31) * 8, wr_oth = wr0 + BUF;
    u32 rd_cur = lds0 + (4 * g + (r16 >> 2)) * ROWB + (lane & 3) * 64;
    u32 tab;
    {
        const u32 u = lane - 7, j = (u * 37) >> 8, i = u - j * 7;
        tab = u < 56 ? ((rank + 1 + i) & 7) * 16 + 2 * j : ks0 + (lane & 7);
    }
    u32 voffXk = OFF_XS - 2048 + wv * 4096 + lane * 16 + half * 2048;
    OPAQUE(voffXk); OPAQUE(tab); OPAQUE(wr0); OPAQUE(wr_oth); OPAQUE(rd_cur);
    const u32 xsd = half ? -2048 : 2048;
    SCHED();
    int sched = PACE_T0 + (((31 - bi) * STAGGER) >> 5), pc = PACE0;
#define WSTORE(base, i) (*(LAS u32x2*)((base) + (i) * 8 * ROWB) = u32x2{cvt2(st[i].x, st[i].y), cvt2(st[i].z, st[i].w)})
#pragma unroll
    for (int i = 0; i < 6; ++i) WSTORE(wr0, i);
    if (send_p >= 0) XIN_PARK();
    SCHED();
    if (wv == POLLW) {
        const u64 t0 = now();
        for (u32 it = 1;; ++it) {
            if (__builtin_amdgcn_ballot_w64(lane < 3 && f0.x < call) == 0) break;
            if ((it & 31) == 0 && give_up(ws, call, t0)) break;
            __builtin_amdgcn_s_sleep(1);
            asm volatile("" ::: "memory");
            f0 = __builtin_amdgcn_raw_buffer_load_b64(rWS, fvo, 0, AUX_SYS);
        }
    }
    asm volatile("s_waitcnt lgkmcnt(0)" ::: "memory");
    BARRIER();
#define KOFF(s, sh) ((u32)__builtin_amdgcn_readlane(tab, (s)) << (sh))
#define XLOAD2(d0, d1, so) do { const u32 so_ = (so); \
        (d0) = __builtin_amdgcn_raw_buffer_load_b128(rWS, voffXk, so_, AUX_SYS); \
        (d1) = __builtin_amdgcn_raw_buffer_load_b128(rWS, voffXk + 1024, so_, AUX_SYS); } while (0)
#define XLOAD(dstv, so) do { const u32 sk_ = (so) + 2048; \
        XLOAD2((dstv)[0], (dstv)[1], sk_); XLOAD2((dstv)[2], (dstv)[3], sk_ + xsd); SCHED(); } while (0)
    u32x4 xr[3][4];
    const u32 pf_off = OFF_XS + grp * 16384 + bi * 512 + wj * 128;
    u32 pfsink = 0, pfsink2 = 0;
    u32x2 fl[2][3], fh[2][3];
#define FADDR(t, q) (rd_cur + (q) * 2 * SLOT + (t) * 8)
#define RL(t, q) (fl[(t) & 1][q] = trread(FADDR(t, q)))
#define RH(t, q) (fh[(t) & 1][q] = trread(FADDR(t, q) + SLOT))
#define MQ(t, i, q) do { mfma(acc[i][t], fl[(t) & 1][q], fh[(t) & 1][q], xr[q][i]); SCHED(); } while (0)
#define MN(t, n, F) do { constexpr int j_ = (n) / 3, i_ = ((t) & 1) ? 3 - j_ : j_, q_ = (j_ & 1) ? (n) % 3 : 2 - (n) % 3; \
        MQ(t, i_, q_); F; SCHED(); } while (0)
#define NOP (void)0
#define TILE6(t, F0, F1, F2, F3, F4, F5) do { \
        MN(t, 0, RL((t) + 1, 2)); MN(t, 1, RH((t) + 1, 2)); MN(t, 2, F0); MN(t, 3, RL((t) + 1, 1)); MN(t, 4, RH((t) + 1, 1)); MN(t, 5, F1); \
        MN(t, 6, RL((t) + 1, 0)); MN(t, 7, RH((t) + 1, 0)); MN(t, 8, F2); MN(t, 9, F3); MN(t, 10, F4); MN(t, 11, F5); } while (0)
    u32 wlo, whi;
#define CA(i) (wlo = cvt2(st[i].x, st[i].y))
#define CB(i) (whi = cvt2(st[i].z, st[i].w))
#define WR(i) (*(LAS u32x2*)(wr_oth + (i) * 8 * ROWB) = u32x2{wlo, whi})
    const bool paced = !warm;
    u64 t_c0 = t_start, t_now = 0;
    bool fast = false;
    u32 pblk = 0;
    u32 cy0 = 0, rt0 = 0;
    bool late = grp;
#define WAIT_LATE(s, n) do { if (late) { \
            const bool act = (voffXk & 1023) - (u32)(s) * 16 < (n) * 16u; \
            const u64 t0 = now(); \
            for (u32 it = 1;; ++it) { \
                const u32x2 f = __builtin_amdgcn_raw_buffer_load_b64(rWS, act ? tab << 7 : OOB, OFF_XFLAG, AUX_SYS); \
                if (__builtin_amdgcn_ballot_w64(act && min(f.x, f.y) < call) == 0) break; \
                if ((it & 31) == 0 && give_up(ws, call, t0)) break; \
                __builtin_amdgcn_s_sleep(2); \
                asm volatile("" ::: "memory"); \
            } \
            __builtin_amdgcn_fence(__ATOMIC_ACQUIRE, "wavefront"); \
            pfsink2 = __builtin_amdgcn_raw_buffer_load_b32(rWS, act ? tab << 15 : OOB, pf_off, AUX_SYS); } SCHED(); } while (0)
    if (grp) {
        pfsink = __builtin_amdgcn_raw_buffer_load_b32(rWS, lane < 3 ? tab << 15 : OOB, pf_off - 16384, AUX_SYS);
        WAIT_LATE(0, 3);
        late = false;
        BARRIER();
    }
    u32x4 vt;
#define KS(s) ((u32)__builtin_amdgcn_readlane(tab, (s) & 63))
    u32 kc[3] = {ks0, ks0 + 1, ks0 + 2};
    u32 fo = (u32)(lane - 3) < 3u ? tab << 7 : OOB, ftmp;
    u32 xmask = 0;
    for (int i = 0; i < 20 - NPER; ++i) xmask |= 1u << (X0 + i * XD + (bi + 3 * grp) % XD);
    int wq = 1, wb = 1, rb = 0;
    u32 kwa = ks0 + 3 + grp, kwb = ks0 + 4 + grp;
    auto period = [&](const int p, auto aux_c, auto first_c) __attribute__((always_inline)) {
        constexpr int aux = decltype(aux_c)::value;
        constexpr bool first = decltype(first_c)::value == 1, lean = decltype(first_c)::value == 2;
        const int s0 = 3 * p;
        u32x2 fg;
        if constexpr (!lean) {
            WAIT_LATE(WLO(p - 1), WLO(p) - WLO(p - 1));
            fg = __builtin_amdgcn_raw_buffer_load_b64(rWS, fo, OFF_XFLAG, AUX_SYS);
        }
        SCHED();
        XLOAD(xr[0], kc[0] << 15);
        RL(0, 0); RH(0, 0); RL(0, 1);
        SCHED();
        XLOAD(xr[1], kc[1] << 15);
        XLOAD(xr[2], kc[2] << 15);
        if (!first && ((xmask >> p) & 1)) {
            const u32 ka = kwa << 21, kb = kwb << 21;
            const u32 h0 = ka + (grp << 20), h1 = grp ? kb : ka + (1u << 20), h2 = kb + (grp << 20);
            st[0] = __builtin_amdgcn_raw_buffer_load_b128(rW, voffW, h0, aux);
            st[1] = __builtin_amdgcn_raw_buffer_load_b128(rW, voffW, h0 + (1u << 19), aux);
            st[2] = __builtin_amdgcn_raw_buffer_load_b128(rW, voffW, h1, aux);
            st[3] = __builtin_amdgcn_raw_buffer_load_b128(rW, voffW, h1 + (1u << 19), aux);
            st[4] = __builtin_amdgcn_raw_buffer_load_b128(rW, voffW, h2, aux);
            st[5] = __builtin_amdgcn_raw_buffer_load_b128(rW, voffW, h2 + (1u << 19), aux);
            if constexpr (lean && TOUCH) {
                const u32 l = voffXk & 1023;
                u32 tv = __builtin_amdgcn_raw_buffer_load_b32(rWS, l == 0 ? OFF_PFLAG + (b * 8 + wv) * 128 : l == 16 ? OFF_AMAX + b * 8 :
                    l == 32 ? OFF_PART + (u32)(bi * 4 + lq) * 131072u + wv * 16384 + (pblk ^ 1) * 8192 : l == 48 ? OFF_GMAX : OOB, 0, AUX_SYS);
                SCHED();
                OPAQUE(tv);
            }
            SCHED();
            const u32 wa = wr0 + wb * BUF;
            ++wq; wb = wb == NBUF - 1 ? 0 : wb + 1;
            kwa = KS(3 * wq + grp); kwb = KS(3 * wq + 1 + grp);
#pragma unroll
            for (int i = 0; i < 6; ++i) WSTORE(wa, i);
            SCHED();
        }
        if (!first && !lean && p == CLK_AT - grp && paced) {
            const u32x4 v = __builtin_amdgcn_raw_buffer_load_b128(rWS, (voffXk & 1023) < 256 ? OFF_CLK + (voffXk & 1023) : OOB, 0, AUX_SYS);
            u32 d = (v.y == call ? v.x - CLK_THR + 65536 : 0) + (v.w == call ? v.z - CLK_THR + 65536 : 0);
            d += dpp<0xB1>(d); d += dpp<0x4E>(d); d += dpp<0x141>(d); d += dpp<0x140>(d);
            const int ds = __builtin_amdgcn_readfirstlane(d);
            fast = ds > 12 * 65536 && (short)ds < 0;
            if (fast && !grp) pc = FAST0;
            SCHED();
        }
        if (!lean && p == PL_AT) {
            st[0] = __builtin_amdgcn_raw_buffer_load_b128(rW, voffW, ((ks0 + 7) * 4 + grp * 2) << 19, aux);
            st[1] = __builtin_amdgcn_raw_buffer_load_b128(rW, voffW, ((ks0 + 7) * 4 + grp * 2 + 1) << 19, aux);
            SCHED();
            const u32 wa = wr0 + (6 * NBUF - 2 * grp) * SLOT;
            WSTORE(wa, 0); WSTORE(wa, 1);
            SCHED();
        }
        const u32 ka = kwa << 21, kb = kwb << 21;
        const u32 h0 = ka + (grp << 20), h1 = grp ? kb : ka + (1u << 20), h2 = kb + (grp << 20);
        st[0] = __builtin_amdgcn_raw_buffer_load_b128(rW, voffW, h0, aux);
        st[1] = __builtin_amdgcn_raw_buffer_load_b128(rW, voffW, h0 + (1u << 19), aux);
        st[2] = __builtin_amdgcn_raw_buffer_load_b128(rW, voffW, h1, aux);
        SCHED();
        asm volatile("s_memrealtime %0" : "+s"(t_now));
        st[3] = __builtin_amdgcn_raw_buffer_load_b128(rW, voffW, h1 + (1u << 19), aux);
        st[4] = __builtin_amdgcn_raw_buffer_load_b128(rW, voffW, h2, aux);
        st[5] = __builtin_amdgcn_raw_buffer_load_b128(rW, voffW, h2 + (1u << 19), aux);
        SCHED();
        if constexpr (!lean) {
        late = __builtin_amdgcn_ballot_w64(fo != OOB && min(fg.x, fg.y) < call) != 0;
        __builtin_amdgcn_fence(__ATOMIC_ACQUIRE, "wavefront");
        const u32 po = late ? OOB : fo << 8 | fo;
        SCHED();
        if constexpr (first) {
            vt = __builtin_amdgcn_raw_buffer_load_b128(rWS, OFF_VOTE + (voffXk & 1023), 0, AUX_SYS);
        }
        SCHED();
        asm volatile("" :: "v"(pfsink), "v"(pfsink2));
        pfsink = __builtin_amdgcn_raw_buffer_load_b32(rWS, po & 0xffff8000u, pf_off, PF_AUX);
        pfsink2 = __builtin_amdgcn_raw_buffer_load_b32(rWS, OOB, 0, 0);
        SCHED();
        if constexpr (!first) asm volatile("s_waitcnt vmcnt(20)" ::: "memory");
        if (p == send_p) {
            const u32 l16 = voffXk & 1023, pk = lds0 + LDS_PARK + wv * 2048 + l16;
            st128x2_sys(wsd + piece_off + wv * 2048, l16, *(LAS u32x4*)pk, *(LAS u32x4*)(pk + 1024));
        }
        }
        asm volatile("s_waitcnt lgkmcnt(0)" : "+s"(t_c0), "+s"(t_now) : "v"(fl[0][0]), "v"(fh[0][0]), "v"(fl[0][1]));
        if (paced) {
            if (!first && !lean && p <= 2 && wv == 4 && b >= 224) {
                const u32 cy = (u32)__builtin_amdgcn_s_memtime(), rt = (u32)now();
                if (p == 2 && (voffXk & 1023) == 0)
                    st64_sys(ws + OFF_CLK + (b & 31) * 8, (u32)((float)(cy - cy0) * 10.f / (float)(rt - rt0)), call);
                cy0 = cy; rt0 = rt;
            }
            const int due = sched + (grp ? pc >> 1 : 0), last = (int)((u32)t_c0 - t_start);
            const int rel = max(due, last + pc * (last + pc - due > FAR ? CATCH_FAR : fast ? FAST_CATCH : CATCH) / 32);
            if ((int)((u32)t_now - t_start) + 10 < rel)
                while ((int)((u32)now() - t_start) < rel) __builtin_amdgcn_s_sleep(1);
        }
        BARRIER();
        RH(0, 1); RL(0, 2); RH(0, 2);
        SCHED();
        asm volatile("s_memrealtime %0" : "+s"(t_c0));
        __builtin_amdgcn_s_setprio(PRIO_C);
        if (!lean && p == send_p + 1 && wv == 4) {
            if ((voffXk & 1023) == 0) st32_sys(wsd + pflag_off, call);
        }
        SCHED();
        const int wo = wb * BUF;
        ++wq; wb = wb == NBUF - 1 ? 0 : wb + 1;
        MQ(0, 0, 0); RL(1, 2); SCHED(); MQ(0, 1, 0); RH(1, 2); SCHED(); MQ(0, 2, 0); MQ(0, 3, 0); RL(1, 1); SCHED();
        MQ(0, 3, 1); RH(1, 1); SCHED(); MQ(0, 2, 1); MQ(0, 1, 1); RL(1, 0); SCHED(); MQ(0, 0, 1); RH(1, 0); SCHED();
        MQ(0, 0, 2); MQ(0, 1, 2); MQ(0, 2, 2); MQ(0, 3, 2);
        TILE6(1, wr_oth = wr0 + wo, kc[0] = KS(s0 + 3), kc[1] = KS(s0 + 4), kc[2] = KS(s0 + 5), kwa = KS(3 * wq + grp), NOP);
        TILE6(2, kwb = KS(3 * wq + 1 + grp), ftmp = (voffXk & 1023) - WLO(p + 1) * 16, fo = tab << 7,
              fo = ftmp < (WLO(p + 2) - WLO(p + 1)) * 16u ? fo : OOB, sched += pc, NOP);
        TILE6(3, pc = p < 7 ? (fast ? FAST0 : PACE0) : p < 15 ? 200 + (int)(((fast ? FAST_RAMP : PACE_RAMP) >> ((p - 7) * 8)) & 255) : PACE1,
              NOP, NOP, NOP, NOP, NOP);
        TILE6(4, NOP, NOP, NOP, CA(0), CB(0), NOP);
        TILE6(5, WR(0), CA(1), CB(1), WR(1), CA(2), NOP);
        TILE6(6, CB(2), WR(2), CA(3), CB(3), WR(3), CA(4));
        MN(7, 0, CB(4)); MN(7, 1, WR(4)); MN(7, 2, CA(5)); MN(7, 3, CB(5)); MN(7, 4, WR(5)); MN(7, 5, NOP); MN(7, 6, NOP); MN(7, 7, NOP);
        asm volatile("s_waitcnt lgkmcnt(0)" ::: "memory");
        __builtin_amdgcn_s_setprio(PRIO_L);
        BARRIER();
        MN(7, 8, NOP); MN(7, 9, NOP); MN(7, 10, NOP); MN(7, 11, NOP);
        rd_cur += rb == NBUF - 1 ? -(NBUF - 1) * BUF : BUF;
        rb = rb == NBUF - 1 ? 0 : rb + 1;
    };
    period(0, IC<W_AUX>(), IC<1>());
    {
        if (__builtin_amdgcn_ballot_w64(vt.y != call || vt.w != call) != 0) {
            const u64 t0 = now();
            for (u32 it = 1;; ++it) {
                vt = __builtin_amdgcn_raw_buffer_load_b128(rWS, OFF_VOTE + (voffXk & 1023), 0, AUX_SYS);
                if (__builtin_amdgcn_ballot_w64(vt.y != call || vt.w != call) == 0) break;
                if ((it & 31) == 0 && give_up(ws, call, t0)) break;
                __builtin_amdgcn_s_sleep(2);
                asm volatile("" ::: "memory");
            }
        }
        u32 d = (vt.x & 0xfffff) + (vt.z & 0xfffff);
        d += dpp<0xB1>(d); d += dpp<0x4E>(d); d += dpp<0x141>(d); d += dpp<0x140>(d);
        const u32 ds = __builtin_amdgcn_readlane(d, 0) + __builtin_amdgcn_readlane(d, 16) +
                       __builtin_amdgcn_readlane(d, 32) + __builtin_amdgcn_readlane(d, 48);
        const int dsum = (int)(ds & 1023) - 128;
        const u32 i01 = (u32)__builtin_amdgcn_readlane((lq & 1) ? vt.z : vt.x, lq >> 1) >> 20;
        const u32 kp = ((dsum > 0) ^ sp ^ i01 ^ half) & 1;
        pblk = ((ds >> 10 > 128) ^ (b >> 5) ^ i01 ^ half) & 1;
        tab += (voffXk & 1023) - 7 * 16 < 56 * 16 ? kp : 0;
        kwa += grp ? kp : 0; kwb += kp;
        fo = fo == OOB ? OOB : tab << 7;
        tid = wv * 64 + ((voffXk & 1023) >> 4);
    }
    int p = 1;
    const int pn = warm ? ALLOC_FIRST : LEAN, pm = warm ? LEAN : NPER;
    for (; p < pn; ++p) period(p, IC<W_AUX>(), IC<0>());
    for (; p < LEAN; ++p) period(p, IC<0>(), IC<0>());
    static_assert(WLO(LEAN) >= 64, "all flags seen");
    WAIT_LATE(WLO(LEAN - 1), 64);
    for (; p < pm; ++p) period(p, IC<W_AUX>(), IC<2>());
    for (; p < NPER; ++p) period(p, IC<0>(), IC<2>());

    asm volatile("s_waitcnt lgkmcnt(0)" : "+s"(t_c0), "+s"(t_now));
    asm volatile("" :: "v"(pfsink), "v"(pfsink2));
    u32x2 gl[3][2], gh[3][2];
    u32x4 xa[NF][2];
    u32 ko[NF];
    static_assert(20 % NBUF == 0, "period 20 in the first buffer, 19 in the last, the last k-step behind it");
    u32 rd_0 = rd_cur - (NPER % NBUF) * BUF;
    OPAQUE(rd_0);
    static_assert(NF == 10, "blocks of 4, 4 and 2 k-steps");
#define GT(u) ((u) < 32 ? ((u) >> 1) & 7 : (u) & 7)
#define GQ(u) ((u) < 32 ? ((u) >> 4) * 4 + ((((u) >> 1) ^ (u)) & 1) * 2 : 8)
#define GADDR(t, q) (((q) < NF - 4 ? rd_cur + (q) * 2 * SLOT : (q) < NF - 1 ? rd_0 + ((q) - (NF - 4)) * 2 * SLOT \
                                                             : rd_cur + (20 - NPER) * BUF) + (t) * 8)
#define GRL(u, k) do { if ((u) < 40) gl[(u) % 3][k] = trread(GADDR(GT(u), GQ(u) + (k))); SCHED(); } while (0)
#define GRH(u, k) do { if ((u) < 40) gh[(u) % 3][k] = trread(GADDR(GT(u), GQ(u) + (k)) + SLOT); SCHED(); } while (0)
#define GM(u, r, i, k) do { mfma(acc[(r) + (i)][GT(u)], gl[(u) % 3][k], gh[(u) % 3][k], xa[GQ(u) + (k)][i]); SCHED(); } while (0)
#define GUNIT(u, r) do { \
        GM(u, r, (u) & 1, 0); GRL((u) + 2, 0); GM(u, r, (u) & 1, 1); GRH((u) + 2, 0); \
        GM(u, r, 1 - ((u) & 1), 1); GRL((u) + 2, 1); GM(u, r, 1 - ((u) & 1), 0); GRH((u) + 2, 1); } while (0)
#define GPASS(r, MID, MID2, MID3, ENDBAR) do { \
        GRL(0, 0); GRH(0, 0); GRL(0, 1); GRH(0, 1); GRL(1, 0); GRH(1, 0); GRL(1, 1); GRH(1, 1);     \
        BARRIER(); \
        __builtin_amdgcn_s_setprio(PRIO_C); \
        _Pragma("unroll") for (int u = 0; u < FLAG_AT; ++u) GUNIT(u, r); \
        MID; \
        _Pragma("unroll") for (int u = FLAG_AT; u < POLL_AT; ++u) GUNIT(u, r); \
        MID2; \
        _Pragma("unroll") for (int u = POLL_AT; u < POLL_AT + 6; ++u) GUNIT(u, r); \
        MID3; \
        _Pragma("unroll") for (int u = POLL_AT + 6; u < 39; ++u) GUNIT(u, r); \
        __builtin_amdgcn_s_setprio(PRIO_L); \
        ENDBAR; \
        GUNIT(39, r); \
        asm volatile("s_nop 7");         \
        _Pragma("unroll") for (int t = 0; t < 8; ++t) asm volatile("" : "+a"(acc[r][t]), "+a"(acc[(r) + 1][t])); \
        SCHED(); } while (0)
#pragma unroll
    for (int a = 0; a < NF; ++a) { ko[a] = KOFF(64 - NF + a, 15) + 2048; XLOAD2(xa[a][0], xa[a][1], ko[a] + xsd); SCHED(); }
    GPASS(2, (void)0, (void)0, (void)0, BARRIER());
#pragma unroll
    for (int a = 0; a < NXA; ++a) { XLOAD2(xa[a][0], xa[a][1], ko[a]); SCHED(); }
    tid = wv * 64 + ((voffXk & 1023) >> 4); lane = tid & 63;
    {
        const u32 vo = OFF_PART + (u32)(bi * 4 + lq) * 131072u + wv * 16384 + (pblk ^ 1) * 8192 + lane * 16;
#pragma unroll
        for (int i = 0; i < 2; ++i)
#pragma unroll
            for (int t = 0; t < 4; ++t) {
                const u32x4 c = __builtin_bit_cast(u32x4, acc[2 + i][2 * t]), d = __builtin_bit_cast(u32x4, acc[2 + i][2 * t + 1]);
                __builtin_amdgcn_raw_buffer_store_b128(u32x4{cvt2(c.x, c.y), cvt2(c.z, c.w), cvt2(d.x, d.y), cvt2(d.z, d.w)},
                                                       rWS, vo + t * 1024 + i * 4096, 0, PART_AUX);
                SCHED();
            }
    }
#pragma unroll
    for (int a = NXA; a < NF; ++a) { XLOAD2(xa[a][0], xa[a][1], ko[a]); SCHED(); }
#define PUBLISH() do { asm volatile("s_waitcnt vmcnt(0)" ::: "memory"); \
               if (lane == 0) st32_sys(ws + OFF_PFLAG + (b * 8 + wv) * 128, call); } while (0)
    const u32 pfo = OFF_PFLAG + ((b ^ 4) * 8 + wv) * 128, vi = OFF_PART + (u32)(bi * 4 + lq) * 131072u + wv * 16384 + pblk * 8192 + lane * 16;
    u32 pfv = 0;
    bool got = false;
    u32x4 in[2][4];
#define FETCH() do { __builtin_amdgcn_fence(__ATOMIC_ACQUIRE, "wavefront"); \
        _Pragma("unroll") for (int i = 0; i < 2; ++i) _Pragma("unroll") for (int t = 0; t < 4; ++t) \
            in[i][t] = __builtin_amdgcn_raw_buffer_load_b128(rWS, vi + t * 1024, i * 4096, FETCH_AUX); } while (0)
    GPASS(0, { _Pragma("unroll") for (int a = FLAG_AT / 16 * 4; a < NF; ++a) asm volatile("" : "+v"(xa[a][0]), "+v"(xa[a][1]));
               if (grp) PUBLISH(); SCHED(); },
          { if (grp) pfv = __builtin_amdgcn_raw_buffer_load_b32(rWS, pfo, 0, AUX_SYS); SCHED(); },
          { if (grp) { got = (u32)__builtin_amdgcn_readfirstlane(pfv) >= call; if (got) FETCH(); } SCHED(); },
          if (!grp) BARRIER());
    if (!grp) PUBLISH();
    SCHED();
    char* wsl = P.ws[0];
    if (wv == 0 && b == GATHER_WG) {
#pragma unroll
        for (int i = 1; i < 8; ++i) wsl = lane == i ? P.ws[i] : wsl;
        OPAQUE(wsl);
    }
    if (!got) {
        GAS u32* pf = (GAS u32*)(ws + pfo);
        if (__hip_atomic_load(pf, __ATOMIC_RELAXED, __HIP_MEMORY_SCOPE_SYSTEM) < call) {
            const u64 t0 = now();
            for (u32 it = 1; __hip_atomic_load(pf, __ATOMIC_RELAXED, __HIP_MEMORY_SCOPE_SYSTEM) < call; ++it) {
                if ((it & 31) == 0 && give_up(ws, call, t0)) break;
                __builtin_amdgcn_s_sleep(1);
            }
        }
        FETCH();
    }
    SCHED();
    float am = 0.f;
    {
#pragma unroll
        for (int i = 0; i < 2; ++i)
#pragma unroll
            for (int t = 0; t < 8; ++t) {
                const u32 lo = in[i][t >> 1][(t & 1) * 2], hi = in[i][t >> 1][(t & 1) * 2 + 1];
                acc[i][t] += f32x4{__builtin_bit_cast(float, lo << 16), __builtin_bit_cast(float, lo & 0xffff0000u),
                                   __builtin_bit_cast(float, hi << 16), __builtin_bit_cast(float, hi & 0xffff0000u)};
#pragma unroll
                for (int k = 0; k < 4; ++k) am = fmaxf(am, fabsf(acc[i][t][k]));
            }
    }
    const u32 amw = wave_max(__builtin_bit_cast(u32, am));
    if (lane == 0) misc[16 + wv] = amw;
    asm volatile("s_waitcnt lgkmcnt(0)" ::: "memory");
    BARRIER();
    u32 gm, bad_call, m;
    if (wv == 0) {
        m = misc[16 + (lane & 7)];
        m = max(m, dpp<0xB1>(m)); m = max(m, dpp<0x4E>(m)); m = max(m, dpp<0x141>(m));
        if (lane == 0) st64_sys(ws + OFF_AMAX + b * 8, m, call);
    }
    if (tid == 64)
        __hip_atomic_store((GAS u64*)(ws + OFF_CNT + b * 64), (__builtin_amdgcn_s_memrealtime() << 32) | call,
                           __ATOMIC_RELAXED, __HIP_MEMORY_SCOPE_AGENT);
    const rsrc_t rO = mk_rsrc(P.out, (u32)MROW * NCOL);
    u32 vo = (u32)(wv * 64 + half * 32 + (lane >> 3)) * NCOL + nt * 128 + (lane & 7) * 16;
    asm volatile("" : "+v"(vo) : "s"(P.out));
    SCHED();
    f32x4 y[4][4];
    {
        constexpr int TROW = 528;
        static_assert(8 * 32 * TROW <= LDS_MISC, "lds");
        const u32 tw = lds0 + (wv * 32 + (lane & 15)) * TROW + (lane >> 4) * 128;
        const u32 tr = lds0 + (wv * 32 + (lane >> 3)) * TROW + (lane & 7) * 64;
#pragma unroll
        for (int i = 0; i < 2; ++i)
#pragma unroll
            for (int t = 0; t < 8; ++t) *(LAS f32x4*)(tw + i * 16 * TROW + t * 16) = acc[i][t];
        asm volatile("" ::: "memory");
#pragma unroll
        for (int j = 0; j < 4; ++j)
#pragma unroll
            for (int k = 0; k < 4; ++k) { y[j][k] = *(LAS f32x4*)(tr + j * 8 * TROW + k * 16); }
#pragma unroll
        for (int j = 0; j < 4; ++j) asm volatile("" : "+v"(y[j][0]), "+v"(y[j][1]), "+v"(y[j][2]), "+v"(y[j][3]));
    }
    SCHED();
    if (wv == 0) {
        const u64 t0 = now();
        if (b == GATHER_WG) {
            u32 oa = OFF_AMAX + lane * 16, oc = oa + 1024;
            for (u32 it = 1;; ++it) {
                const u32x4 a = __builtin_amdgcn_raw_buffer_load_b128(rWS, oa, 0, AUX_SYS);
                const u32x4 c = __builtin_amdgcn_raw_buffer_load_b128(rWS, oc, 0, AUX_SYS);
                if (a.y == call && a.w == call) { m = max(m, max(a.x, a.z)); oa = OOB; }
                if (c.y == call && c.w == call) { m = max(m, max(c.x, c.z)); oc = OOB; }
                if (__builtin_amdgcn_ballot_w64((oa & oc) != OOB) == 0) break;
                if ((it & 31) == 0 && give_up(ws, call, t0)) break;
                asm volatile("" ::: "memory");
            }
            m = wave_max(m);
            if (lane < 8) st64_sys(wsl + OFF_GMAX + rank * 8, m, call);
        }
        const u32 og = lane < 8 ? OFF_GMAX + lane * 8 : lane == 8 ? OFF_TMO : OOB;
        const bool gact = lane < 8;
        u32x2 gv;
        bool bad = false;
        for (u32 it = 1;; ++it) {
            gv = __builtin_amdgcn_raw_buffer_load_b64(rWS, og, 0, AUX_SYS);
            if (__builtin_amdgcn_ballot_w64(gact && gv.y != call) == 0) break;
            if ((it & 31) == 0 && give_up(ws, call, t0)) { bad = true; break; }
            __builtin_amdgcn_s_sleep(AMAX_SLEEP);
            asm volatile("" ::: "memory");
        }
        bad = bad || (u32)__builtin_amdgcn_readlane(gv.x, 8) == call;
        m = gact ? gv.x : 0;
        m = max(m, dpp<0xB1>(m)); m = max(m, dpp<0x4E>(m)); m = max(m, dpp<0x141>(m));
        if (lane == 0) misc[32] = bad ? 0xffffffffu : m;
        asm volatile("s_waitcnt lgkmcnt(0)" ::: "memory");
    }
    BARRIER();
    gm = misc[32];
    bad_call = gm == 0xffffffffu;
    {
        const float amax = __builtin_bit_cast(float, gm);
        const float inv = amax > 0.f ? 127.0f / amax : 0.f;
        const float lo = 12582912.0f - (bad_call ? 128.f : 127.f), hi = bad_call ? lo : 12582912.0f + 127.f;
#pragma unroll
        for (int j = 0; j < 4; ++j) {
            u32x4 q;
#pragma unroll
            for (int t = 0; t < 4; ++t) {
                u32 e[4];
                const f32x4 v = __builtin_elementwise_fma(y[j][t], f32x4{inv, inv, inv, inv},
                                                          f32x4{12582912.0f, 12582912.0f, 12582912.0f, 12582912.0f});
#pragma unroll
                for (int k = 0; k < 4; ++k) e[k] = __builtin_bit_cast(u32, __builtin_amdgcn_fmed3f(v[k], lo, hi));
                q[t] = __builtin_amdgcn_perm(__builtin_amdgcn_perm(e[3], e[2], 0x0c0c0400u),
                                             __builtin_amdgcn_perm(e[1], e[0], 0x0c0c0400u), 0x05040100u);
            }
            __builtin_amdgcn_raw_buffer_store_b128(q, rO, vo + j * 8 * NCOL, 0, OUT_AUX);
            SCHED();
        }
    }
}

extern "C" void kernel_launch(int rank, int nranks, ncclComm_t comm,
                              hipStream_t stream,
                              void* const* d_in, const int* in_sizes, int n_in,
                              void* d_out, int out_size,
                              const int* in_dtypes, int out_dtype,
                              void* const* d_out_all,
                              void* const* d_ws_all, size_t ws_bytes) {
    if (nranks != NRANK || (unsigned)rank >= (unsigned)NRANK || ws_bytes < WS_USED + (16u << 20)) {
        fprintf(stderr, "unexpected configuration\n");
        return;
    }
    HIP_CHECK(hipFuncSetAttribute((const void*)a2a_gemm_q8, hipFuncAttributeMaxDynamicSharedMemorySize, LDS_BYTES));
    Params P;
    P.x = (const float*)d_in[0];
    P.w = (const float*)d_in[1];
    P.out = (char*)d_out;
    for (int p = 0; p < NRANK; ++p) P.ws[p] = (char*)d_ws_all[(rank + p) % NRANK];
    P.rank = rank;
    P.pad = 0;
    hipLaunchKernelGGL(a2a_gemm_q8, dim3(NWG), dim3(NTHR), LDS_BYTES, stream, P);
    HIP_CHECK(hipGetLastError());
}
```
